```python
import numpy as np

kernel_name = "gpu_a2a_gemm_m4096_k4096_n4096_w8a8_dqsilu"


ABSMAX_RTOL = 0.001

COMPUTE_DTYPE = "int8"
EPILOGUE = "dequant_silu"
IN_QUANT = "int8"
OUT_QUANT = None
IN_DTYPES = {"x": "int8", "w_mat": "int8",
             "scale_x": "float32", "scale_w": "float32"}
WORLD_SIZE = 8

def setup_inputs():

    x = np.random.randint(-127, 128, size=(4096, 4096)).astype(np.int8)
    w_mat = np.random.randint(-127, 128, size=(4096, 4096)).astype(np.int8)
    scale_x = (0.02 * (1.0 + np.random.rand(1))).astype(np.float32)
    scale_w = (0.02 * (1.0 + np.random.rand(1))).astype(np.float32)
    return {"x": x, "w_mat": w_mat, "scale_x": scale_x, "scale_w": scale_w}

def reference(x, w_mat, scale_x, scale_w):
    acc = x.astype(np.float64) @ w_mat.astype(np.float64)
    y = (acc.astype(np.float32) * (scale_x * scale_w)).astype(np.float32)
    return (y / (1.0 + np.exp(-np.clip(y, -60.0, 60.0)))).astype(np.float32)
```

```cpp
#include <hip/hip_runtime.h>
#include <rccl/rccl.h>
#include <cstdio>
#include <cstdint>

#define HIP_CHECK(expr) do { hipError_t _e = (expr); if (_e != hipSuccess) { \
    fprintf(stderr, "HIP error %s at %s:%d\n", hipGetErrorString(_e), __FILE__, __LINE__); \
    return; } } while (0)

namespace {

typedef int v2i __attribute__((ext_vector_type(2)));
typedef int v4i __attribute__((ext_vector_type(4)));
typedef int v16i __attribute__((ext_vector_type(16)));
typedef unsigned v4u __attribute__((ext_vector_type(4)));
#define LAS __attribute__((address_space(3)))
#define GAS __attribute__((address_space(1)))

constexpr int WORLD = 8;
constexpr int NN = 4096;
constexpr int KC = 512;
constexpr int NWG = 256;
constexpr int NTHR = 512;
constexpr int NSLOT = 16;
constexpr int DB = 12;
constexpr int NSTAGE = 32;

constexpr size_t CTR_OFF  = 0;
constexpr size_t TMO_OFF  = 32768;
constexpr size_t FLAG_OFF = 65536;
constexpr size_t STG_OFF  = 1u << 20;
constexpr size_t STG_BYTES = 2u << 20;
constexpr size_t WS_NEED = STG_OFF + 2 * STG_BYTES;
constexpr unsigned long long DEADLINE_TICKS = 30000000ull;

constexpr int LDS_RING = NSLOT * 8192;
constexpr int LDS_BYTES = LDS_RING + 64;

struct Params {
    const signed char* x;
    const signed char* w;
    const float* sx;
    const float* sw;
    float* out;
    char* ws[WORLD];
    int rank;
};

__device__ __forceinline__ unsigned long long rt_clock() { return __builtin_amdgcn_s_memrealtime(); }

__device__ __forceinline__ int rfl(int v) { return __builtin_amdgcn_readfirstlane(v); }

__device__ __forceinline__ void unit_info(int u, int rank, int& p, int& ks) {
    if (u < 16) { p = rank; ks = u; }
    else {
        int v = u - 16;
        ks = (v * 9363) >> 16;
        int i = v - ks * 7;
        p = (rank + 1 + i) & 7;
    }
}

__device__ __forceinline__ void dma16(const char* sbase, unsigned voff, unsigned lds_dst) {
    unsigned keep;
    asm volatile("s_mov_b32 %0, m0\n\t"
                 "s_mov_b32 m0, %3\n\t"
                 "s_nop 0\n\t"
                 "global_load_lds_dwordx4 %1, %2\n\t"
                 "s_mov_b32 m0, %0"
                 : "=&s"(keep) : "v"(voff), "s"(sbase), "s"(lds_dst) : "memory");
}

__device__ __forceinline__ void aload(v4i& dst, const char* sbase, unsigned voff) {
    asm volatile("global_load_dwordx4 %0, %1, %2 sc0 sc1" : "+v"(dst) : "v"(voff), "s"(sbase) : "memory");
}

__device__ __forceinline__ v4i trread(unsigned addr0) {
    v2i lo = __builtin_amdgcn_ds_read_tr8_b64_v2i32((LAS v2i*)(uintptr_t)addr0);
    v2i hi = __builtin_amdgcn_ds_read_tr8_b64_v2i32((LAS v2i*)(uintptr_t)(addr0 + 512));
    v4i r; r.x = lo.x; r.y = lo.y; r.z = hi.x; r.w = hi.y;
    return r;
}

#define WAITV(n) asm volatile("s_waitcnt vmcnt(" #n ")" ::: "memory")

__global__ void __launch_bounds__(NTHR, 2) fused_kernel(Params P) {
    extern __shared__ __attribute__((aligned(16))) char smem[];
    const int tid = threadIdx.x;
    const int lane = tid & 63;
    const int wave = rfl(tid >> 6);
    const int bid = blockIdx.x;
    const int rank = P.rank;
    char* const myws = P.ws[rank];

    unsigned* ctr = (unsigned*)(myws + CTR_OFF + (size_t)bid * 128);
    const unsigned call = (unsigned)rfl((int)__hip_atomic_load(ctr, __ATOMIC_RELAXED, __HIP_MEMORY_SCOPE_AGENT)) + 1u;
    const unsigned par = call & 1u;
    volatile LAS unsigned* lds_bad = (volatile LAS unsigned*)(uintptr_t)((unsigned)(uintptr_t)(LAS char*)smem + LDS_RING);

    if (wave >= 5) {
        const int sw = wave - 5;
        v4u val[3];
        char* dptr[3];
        unsigned* fptr[3];
#pragma unroll
        for (int c = 0; c < 3; ++c) {
            int i = sw + 3 * c;
            if (i < 7) {
                int f = i * 256 + bid;
                int ks = f / 112;
                int rem = f - ks * 112;
                int di = rem >> 4, rb = rem & 15;
                unsigned d = (unsigned)(rank + 1 + di) & 7u;
                const signed char* src = P.x + (size_t)(d * KC + rb * 32 + (lane & 31)) * KC + ks * 32 + (lane >> 5) * 16;
                val[c] = *(const v4u*)src;
                char* dws = P.ws[d];
                dptr[c] = dws + STG_OFF + (size_t)par * STG_BYTES + (size_t)((rank * 16 + rb) * 16 + ks) * 1024 + lane * 16;
                fptr[c] = (unsigned*)(dws + FLAG_OFF + (size_t)(((rb * 16 + ks) * 8) + rank) * 4);
            }
        }
#pragma unroll
        for (int c = 0; c < 3; ++c) {
            if (sw + 3 * c < 7) {
                asm volatile("global_store_dwordx4 %0, %1, off sc0 sc1\n\ts_nop 1" :: "v"(dptr[c]), "v"(val[c]) : "memory");
            }
        }
        if (sw == 0) { WAITV(2); } else { WAITV(1); }
        if (lane == 0) __hip_atomic_store(fptr[0], call, __ATOMIC_RELAXED, __HIP_MEMORY_SCOPE_SYSTEM);
        if (sw == 0) { WAITV(2); } else { WAITV(1); }
        if (lane == 0) __hip_atomic_store(fptr[1], call, __ATOMIC_RELAXED, __HIP_MEMORY_SCOPE_SYSTEM);
        if (sw == 0) {
            WAITV(2);
            if (lane == 0) __hip_atomic_store(fptr[2], call, __ATOMIC_RELAXED, __HIP_MEMORY_SCOPE_SYSTEM);
        }
        return;
    }

    const int xcd = bid & 7;
    const int nb = xcd * 8 + ((bid >> 3) & 7);
    const int mb = bid >> 6;

    if (wave == 4) {
        if (lane == 0) *lds_bad = 0u;
        const unsigned* fl = (const unsigned*)(myws + FLAG_OFF) + (size_t)(mb * 4) * 128 + lane * 8;
        unsigned* tmo = (unsigned*)(myws + TMO_OFF);
        int ready = 0;
        bool dead = false;
        const unsigned long long t0 = rt_clock();
        for (int t = 0; t < NSTAGE; ++t) {
            int need = 4 * t + 4; if (need > 112) need = 112;
            while (ready < need && !dead) {
                v4u f0, f1;
                asm volatile("global_load_dwordx4 %0, %2, off sc0 sc1\n\t"
                             "global_load_dwordx4 %1, %2, off offset:16 sc0 sc1\n\t"
                             "s_waitcnt vmcnt(0)"
                             : "=&v"(f0), "=&v"(f1) : "v"(fl) : "memory");
                unsigned m = 0;
                m |= ((int)(f0.x - call) >= 0) ? 1u : 0u;
                m |= ((int)(f0.y - call) >= 0) ? 2u : 0u;
                m |= ((int)(f0.z - call) >= 0) ? 4u : 0u;
                m |= ((int)(f0.w - call) >= 0) ? 8u : 0u;
                m |= ((int)(f1.x - call) >= 0) ? 16u : 0u;
                m |= ((int)(f1.y - call) >= 0) ? 32u : 0u;
                m |= ((int)(f1.z - call) >= 0) ? 64u : 0u;
                m |= ((int)(f1.w - call) >= 0) ? 128u : 0u;
                unsigned mm = (m | (m << 8)) >> ((rank + 1) & 7);
                mm = (mm & 0x7fu) | 0x100u;
                int lead = __builtin_ctz(~mm);
                int l2 = min(lead, __shfl_xor(lead, 16));
                l2 = min(l2, __shfl_xor(l2, 32));
                unsigned long long full = __ballot(l2 >= 7);
                unsigned fm = (unsigned)full & 0xffffu;
                int nfull = __builtin_ctz(~fm);
                int part = (nfull < 16) ? __builtin_amdgcn_readlane(l2, nfull) : 0;
                ready = rfl(nfull * 7 + part);
                if (ready < need) {
                    unsigned tv = (unsigned)rfl((int)__hip_atomic_load(tmo, __ATOMIC_RELAXED, __HIP_MEMORY_SCOPE_AGENT));
                    if (tv == call || rt_clock() - t0 > DEADLINE_TICKS) {
                        if (lane == 0) { __hip_atomic_store(tmo, call, __ATOMIC_RELAXED, __HIP_MEMORY_SCOPE_AGENT); *lds_bad = 1u; }
                        dead = true;
                    }
                    __builtin_amdgcn_s_sleep(1);
                }
            }
            asm volatile("s_waitcnt lgkmcnt(0)" ::: "memory");
            __builtin_amdgcn_s_barrier();
        }
        __builtin_amdgcn_s_barrier();
        return;
    }

    const int rb = mb * 4 + wave;
    const unsigned lds0 = (unsigned)(uintptr_t)(LAS char*)smem;
    const char* const wbase = (const char*)P.w + nb * 64;
    const unsigned w_voff = (unsigned)((lane >> 2) * NN + 16 * ((lane & 3) ^ (((lane >> 4) & 1) << 1)));
    const unsigned a_voff_loc = (unsigned)((lane & 31) * KC + (lane >> 5) * 16);
    const unsigned a_voff_rem = (unsigned)(lane * 16);
    const char* const a_loc_base = (const char*)P.x + (size_t)(rank * KC + rb * 32) * KC;
    const char* const a_rem_base = myws + STG_OFF + (size_t)par * STG_BYTES + (size_t)rb * 16 * 1024;
    const int g = lane >> 4, q = (lane & 15) >> 1, pp = lane & 1;
    const unsigned tr_row = (unsigned)((16 * (g >> 1) + q) * 64 + (g & 1) * 16 + pp * 8);
    const unsigned tr0 = lds0 + tr_row + (unsigned)(((q >> 2) & 1) * 32);
    const unsigned tr1 = lds0 + tr_row + (unsigned)((((q >> 2) & 1) ^ 1) * 32);

    auto issueB = [&](int s) {
        int p, ks; unit_info(4 * s + wave, rank, p, ks);
        const char* sb = wbase + (size_t)(p * KC + ks * 32) * NN;
        unsigned dst = lds0 + (unsigned)((s & (NSLOT - 1)) * 8192 + wave * 2048);
        dma16(sb, w_voff, dst);
        dma16(sb + 16 * NN, w_voff, dst + 1024);
    };
    auto issueA = [&](v4i& dst, int u) {
        int p, ks; unit_info(u, rank, p, ks);
        const char* sb = (u < 16) ? a_loc_base + ks * 32 : a_rem_base + (size_t)(p * 256 + ks) * 1024;
        aload(dst, sb, (u < 16) ? a_voff_loc : a_voff_rem);
    };

    v16i acc0 = {}, acc1 = {};
    v4i A[4][4];
#pragma unroll
    for (int s = 0; s < 4; ++s)
#pragma unroll
        for (int j = 0; j < 4; ++j) A[s][j] = (v4i){0, 0, 0, 0};

    issueB(0);
#pragma unroll
    for (int s = 0; s < 4; ++s) {
        issueB(s + 1);
#pragma unroll
        for (int j = 0; j < 4; ++j) issueA(A[s][j], 4 * s + j);
    }
#pragma unroll
    for (int s = 5; s < DB; ++s) issueB(s);
    WAITV(32);
    asm volatile("" : "+v"(A[0][0]), "+v"(A[0][1]), "+v"(A[0][2]), "+v"(A[0][3]));

    v4i b0, b1;
    for (int T = 0; T < NSTAGE / 4; ++T) {
#pragma unroll
        for (int s4 = 0; s4 < 4; ++s4) {
            const int t = T * 4 + s4;
            __builtin_amdgcn_s_barrier();
            asm volatile("" ::: "memory");
            const unsigned so = (unsigned)((t & (NSLOT - 1)) * 8192);
            const unsigned sn = (unsigned)(((t + 1) & (NSLOT - 1)) * 8192);
            if (t == 0) { b0 = trread(tr0 + so); b1 = trread(tr1 + so); }
            if (t + DB < NSTAGE) issueB(t + DB);
#pragma unroll
            for (int j = 0; j < 4; ++j) {
                v4i n0, n1;
                if (j < 3) { n0 = trread(tr0 + so + (j + 1) * 2048); n1 = trread(tr1 + so + (j + 1) * 2048); }
                else       { n0 = trread(tr0 + sn); n1 = trread(tr1 + sn); }
                acc0 = __builtin_amdgcn_mfma_i32_32x32x32_i8(A[s4][j], b0, acc0, 0, 0, 0);
                acc1 = __builtin_amdgcn_mfma_i32_32x32x32_i8(A[s4][j], b1, acc1, 0, 0, 0);
                if (t + 4 < NSTAGE) issueA(A[s4][j], 4 * (t + 4) + j);
                b0 = n0; b1 = n1;
            }
            if (t == 0)      { WAITV(32); }
            else if (t == 1) { WAITV(32); }
            else if (t == 2) { WAITV(32); }
            else if (t <= 19) { WAITV(18); }
            else if (t == 20) { WAITV(16); }
            else if (t == 21) { WAITV(14); }
            else if (t <= 27) { WAITV(12); }
            else if (t == 28) { WAITV(8); }
            else if (t == 29) { WAITV(4); }
            else { WAITV(0); }
            asm volatile("" : "+v"(A[(s4 + 1) & 3][0]), "+v"(A[(s4 + 1) & 3][1]), "+v"(A[(s4 + 1) & 3][2]), "+v"(A[(s4 + 1) & 3][3]));
        }
    }
    __builtin_amdgcn_s_barrier();
    asm volatile("" ::: "memory");
    const bool bad = (*lds_bad) != 0u;

    const float scale = P.sx[0] * P.sw[0];
    float* optr = P.out + (size_t)(rb * 32 + 4 * (lane >> 5)) * NN + nb * 64 + (lane & 31);
#pragma unroll
    for (int r = 0; r < 16; ++r) {
        const int row = (r & 3) + 8 * (r >> 2);
        float y0 = (float)acc0[r] * scale;
        float y1 = (float)acc1[r] * scale;
        float c0 = fminf(fmaxf(y0, -60.f), 60.f);
        float c1 = fminf(fmaxf(y1, -60.f), 60.f);
        float o0 = y0 * __builtin_amdgcn_rcpf(1.0f + __expf(-c0));
        float o1 = y1 * __builtin_amdgcn_rcpf(1.0f + __expf(-c1));
        if (bad) { o0 = __builtin_nanf(""); o1 = o0; }
        optr[(size_t)row * NN] = o0;
        optr[(size_t)row * NN + 32] = o1;
    }
    if (wave == 0 && lane == 0) __hip_atomic_store(ctr, call, __ATOMIC_RELAXED, __HIP_MEMORY_SCOPE_AGENT);
}

}

extern "C" void kernel_launch(int rank, int nranks, ncclComm_t comm,
                              hipStream_t stream,
                              void* const* d_in, const int* in_sizes, int n_in,
                              void* d_out, int out_size,
                              const int* in_dtypes, int out_dtype,
                              void* const* d_out_all,
                              void* const* d_ws_all, size_t ws_bytes) {
    if (nranks != WORLD || ws_bytes < WS_NEED || (unsigned)rank >= (unsigned)WORLD) {
        fprintf(stderr, "unsupported configuration\n");
        return;
    }
    static bool attr_set[WORLD] = {};
    if (!attr_set[rank]) {
        HIP_CHECK(hipFuncSetAttribute((const void*)fused_kernel, hipFuncAttributeMaxDynamicSharedMemorySize, LDS_BYTES));
        attr_set[rank] = true;
    }
    Params P;
    P.x = (const signed char*)d_in[0];
    P.w = (const signed char*)d_in[1];
    P.sx = (const float*)d_in[2];
    P.sw = (const float*)d_in[3];
    P.out = (float*)d_out;
    for (int p = 0; p < WORLD; ++p) P.ws[p] = (char*)d_ws_all[p];
    P.rank = rank;
    hipLaunchKernelGGL(fused_kernel, dim3(NWG), dim3(NTHR), LDS_BYTES, stream, P);
}
```

```cpp
#include <hip/hip_runtime.h>
#include <rccl/rccl.h>
#include <cstdio>
#include <cstdint>

#define HIP_CHECK(expr) do { hipError_t _e = (expr); if (_e != hipSuccess) { \
    fprintf(stderr, "HIP error %s at %s:%d\n", hipGetErrorString(_e), __FILE__, __LINE__); \
    return; } } while (0)

#define A_LD_MOD "sc0 sc1"
#define ST_MOD "sc0 sc1"

#define W_TR 1
#define NO_APF 1
#define NO_TOUCH 1
#define ADJ 100
namespace {

typedef int v2i __attribute__((ext_vector_type(2)));
typedef int v4i __attribute__((ext_vector_type(4)));
typedef int v16i __attribute__((ext_vector_type(16)));
typedef unsigned v4u __attribute__((ext_vector_type(4)));
#define LAS __attribute__((address_space(3)))
#define GAS __attribute__((address_space(1)))

constexpr int WORLD = 8;
constexpr int NN = 4096;
constexpr int KC = 512;
constexpr int NWG = 256;
constexpr int NLW = 4;
constexpr int NWAVE = 12;
constexpr int NTHR = NWAVE * 64;
#define NB_ 8
#define PACE_ 66
#define LOAD_DELAY 0
#define APF_MOD ""
#define OUT_POL 2
#define ADJ2 0
#define WARM_GUARD 0
#define REGPF 0
#define THR_G 0
#define THR_N 0
#define PACE_T0 100
#define RELAY_ID 8
#define W2 0
#define WARM_FIRST 2
#define WARM_DELAY 150
#define WARM_STEP 80
#define LSLEEP 1
#define POLL_MOD "sc0 sc1"
#define POLL_SLEEP_S "2"
constexpr int NB = NB_;
constexpr int DX = 4;
constexpr int NSTAGE = 32;
static_assert(NB == 8, "the K loop is unrolled over 8 ring slots");

constexpr size_t CTR_OFF  = 0;
constexpr size_t TMO_OFF  = 32768;
constexpr size_t FLAG_OFF = 65536;
constexpr size_t MBX_OFF  = 131072;
constexpr int    NSTG = 4;
constexpr size_t STG_OFF  = 1u << 20;
constexpr size_t STG_BYTES = 2u << 20;
constexpr size_t WS_NEED = STG_OFF + NSTG * STG_BYTES;
constexpr unsigned long long DEADLINE_TICKS = 30000000ull;

constexpr int LDS_B = 0;
constexpr int LDS_SND = NB * 16384;
constexpr int LDS_SYNC = LDS_SND + 8192;
constexpr int LDS_BYTES = LDS_SYNC + 64;
static_assert(LDS_BYTES <= 163840, "LDS");

struct Params {
    const signed char* x;
    const signed char* w;
    const float* sx;
    const float* sw;
    float* out;
    char* ws[WORLD];
    int rank;
};

__device__ __forceinline__ unsigned long long rt_clock() { return __builtin_amdgcn_s_memrealtime(); }

__device__ __forceinline__ int rfl(int v) { return __builtin_amdgcn_readfirstlane(v); }

__device__ __forceinline__ void stage_info(int s, int rank, int mb, int& p, int& kq) {
    if (s < 4) { p = rank; kq = s; }
    else {
        int v = s - 4;
        kq = (v * 37) >> 8;
        int j = v - kq * 7;
        int pos = j + ((j >= ((mb + 1) & 7)) ? 1 : 0);
        int di = (mb - pos) & 7;
        p = (rank - 1 - di) & 7;
    }
}

#define WAITV(n) asm volatile("s_waitcnt vmcnt(" #n ")" ::: "memory")

__device__ __forceinline__ int lds_min4(unsigned addr) {
    v4i v = *(volatile LAS v4i*)(uintptr_t)addr;
    return rfl(min(min(v.x, v.y), min(v.z, v.w)));
}
__device__ __forceinline__ void lds_put(unsigned addr, int v) { *(volatile LAS int*)(uintptr_t)addr = v; }
__device__ __forceinline__ int lds_get(unsigned addr) { return rfl(*(volatile LAS int*)(uintptr_t)addr); }

__global__ void __launch_bounds__(NTHR, 3) fused_kernel(Params P) {
    extern __shared__ __attribute__((aligned(16))) char smem[];
    const int tid = threadIdx.x;
    const int lane = tid & 63;
    const int wave = rfl(tid >> 6);
    const int bid = blockIdx.x;
    const int rank = P.rank;
    char* const myws = P.ws[rank];
    const unsigned long long t_start = rt_clock();
    const unsigned lds0 = (unsigned)(uintptr_t)(LAS char*)smem;
    const unsigned L_LANDB = lds0 + LDS_SYNC, L_CONS = L_LANDB + 32, L_READY = L_LANDB + 48, L_BAD = L_LANDB + 52;

    unsigned* ctr = (unsigned*)(myws + CTR_OFF + (size_t)bid * 128);
    const unsigned call = *(const __attribute__((address_space(4))) unsigned*)(uintptr_t)ctr + 1u;
    const int xcd = bid & 7;
    const int nb = xcd * 4 + ((bid >> 3) & 3);
    const int mb = bid >> 5;

    if (wave < 2) {
        const int idx = bid * 2 + wave;
        int kq, rg, stg; unsigned d;
        if (idx < 64) { kq = idx >> 4; rg = idx & 15; d = (unsigned)rank; stg = kq; }
        else {
            const int r = idx - 64, n = r / 7, di = r - n * 7;
            const int pos = (n >> 1) & 7, mbd = (pos + di) & 7;
            kq = n >> 4;
            rg = mbd * 2 + (n & 1);
            d = (unsigned)(rank + 1 + di) & 7u;
            stg = 4 + kq * 7 + pos - (pos > ((mbd + 1) & 7) ? 1 : 0);
        }
        v4u val[4];
        const signed char* src = P.x + (size_t)(d * KC + rg * 32 + (lane >> 3)) * KC + kq * 128 + (lane & 7) * 16;
#pragma unroll
        for (int i = 0; i < 4; ++i) val[i] = *(const v4u*)(src + (size_t)i * 8 * KC);
        char* dws = P.ws[d];
        char* dst = dws + STG_OFF + (size_t)(call & (NSTG - 1)) * STG_BYTES + (size_t)(((kq * 8 + rank) * 16 + rg) * 4096) + lane * 16;
        unsigned* flg = (unsigned*)(dws + FLAG_OFF) + (rg >> 1) * 64 + stg * 2 + (rg & 1);
        __builtin_amdgcn_s_barrier();
        const unsigned scr = lds0 + LDS_SND + (unsigned)wave * 4096u;
#pragma unroll
        for (int i = 0; i < 4; ++i)
            *(LAS v4u*)(uintptr_t)(scr + (unsigned)((8 * i + (lane >> 3)) * 128 + 16 * ((lane & 7) ^ ((4 * i + (lane >> 4)) & 7)))) = val[i];
        asm volatile("s_waitcnt lgkmcnt(0)" ::: "memory");
#pragma unroll
        for (int u = 0; u < 4; ++u)
            val[u] = *(LAS v4u*)(uintptr_t)(scr + (unsigned)((lane & 31) * 128 + 16 * ((2 * u + (lane >> 5)) ^ ((lane >> 1) & 7))));
        const long long due = idx < 64 ? 0 : (long long)(PACE_T0 + (((idx - 64) * PACE_) >> 6));
        while ((long long)(rt_clock() - t_start) < due) __builtin_amdgcn_s_sleep(1);
        asm volatile("global_store_dwordx4 %0, %1, off " ST_MOD "\n\t"
                     "global_store_dwordx4 %0, %2, off offset:1024 " ST_MOD "\n\t"
                     "global_store_dwordx4 %0, %3, off offset:2048 " ST_MOD "\n\t"
                     "global_store_dwordx4 %0, %4, off offset:3072 " ST_MOD "\n\t"
                     "s_waitcnt vmcnt(0)" :: "v"(dst), "v"(val[0]), "v"(val[1]), "v"(val[2]), "v"(val[3]) : "memory");
        if (lane == 0) __hip_atomic_store(flg, call, __ATOMIC_RELAXED, __HIP_MEMORY_SCOPE_SYSTEM);
        if (lane == 0) lds_put(L_LANDB + 56 + wave * 4, 1);
        return;
    }

    if (wave == 2) {
        int ready = 0;
        if (lane < 16) lds_put(L_LANDB + lane * 4, lane == 12 ? (ready == NSTAGE ? 1000 : 1 - DX) : 0);
        asm volatile("s_waitcnt lgkmcnt(0)" ::: "memory");
        __builtin_amdgcn_s_barrier();
        const bool relay = (bid & 31) == (mb | RELAY_ID);
        const unsigned* fl = relay ? (const unsigned*)(myws + FLAG_OFF) + mb * 64 + lane : (const unsigned*)(myws + MBX_OFF + (size_t)bid * 128);
        unsigned* mbx = (unsigned*)(myws + MBX_OFF + (size_t)(mb * 32 + (lane & 31)) * 128);
        unsigned* tmo = (unsigned*)(myws + TMO_OFF);
#define PTR_(x) ""
        while (ready < NSTAGE) {
            unsigned long long m, tmp;
            int iters = 256, t, t2;
            int rdy = rfl(ready);
            if (relay) {
                asm volatile(
                    "global_load_dword v100, %[addr], off " POLL_MOD "\n\t"
                    "s_sleep " POLL_SLEEP_S "\n\t"
                    "global_load_dword v101, %[addr], off " POLL_MOD "\n\t"
                    "s_sleep " POLL_SLEEP_S "\n\t"
                    "global_load_dword v102, %[addr], off " POLL_MOD "\n\t"
                    "s_sleep " POLL_SLEEP_S "\n\t"
                    "global_load_dword v103, %[addr], off " POLL_MOD "\n\t"
                    "s_sleep " POLL_SLEEP_S "\n\t"
                    "global_load_dword v104, %[addr], off " POLL_MOD "\n\t"
                    "s_sleep " POLL_SLEEP_S "\n\t"
                    "global_load_dword v105, %[addr], off " POLL_MOD "\n\t"
                    "s_sleep " POLL_SLEEP_S "\n\t"
                    "1:\n\t"
                    "s_waitcnt vmcnt(5)\n\t"
                    "v_subrev_u32 v110, %[call], v100\n\t"
                    "v_cmp_le_i32 %[m], 0, v110\n\t"
                    "global_load_dword v100, %[addr], off " POLL_MOD "\n\t"
                    "s_nop 2\n\t"
                    "s_lshr_b64 %[tmp], %[m], 1\n\t"
                    "s_and_b64 %[m], %[m], %[tmp]\n\t"
                    "s_or_b64 %[m], %[m], %[odd]\n\t"
                    "s_not_b64 %[m], %[m]\n\t"
                    "s_ff1_i32_b64 %[t], %[m]\n\t"
                    "s_lshr_b32 %[t], %[t], 1\n\t"
                    "s_min_u32 %[t], %[t], 32\n\t"
                    "s_cmp_le_u32 %[t], %[rdy]\n\t"
                    "s_cbranch_scc1 100f\n\t"
                    "s_mov_b32 %[rdy], %[t]\n\t"
                    "s_add_u32 %[t2], %[t], %[c64]\n\t"
                    "v_mov_b32 v112, %[t2]\n\t"
                    "global_store_dword %[mbx], v112, off sc1\n\t"
                    "s_sub_u32 %[t2], %[t], 3\n\t"
                    "s_cmp_ge_u32 %[t], 32\n\t"
                    "s_cselect_b32 %[t2], 1000, %[t2]\n\t"
                    "v_mov_b32 v111, %[t2]\n\t"
                    "ds_write_b32 %[lrdy], v111\n\t"
                    PTR_("s_memrealtime s[98:99]")
                    PTR_("s_min_u32 m0, %[t], 32")
                    PTR_("s_waitcnt lgkmcnt(0)")
                    PTR_("v_writelane_b32 %[tl], s98, m0")
                    PTR_("s_cmp_ge_u32 %[t], 32")
                    "s_cbranch_scc1 2f\n\t"
                    "100:\n\t"
                    "s_sleep " POLL_SLEEP_S "\n\t"
                    "s_waitcnt vmcnt(5)\n\t"
                    "v_subrev_u32 v110, %[call], v101\n\t"
                    "v_cmp_le_i32 %[m], 0, v110\n\t"
                    "global_load_dword v101, %[addr], off " POLL_MOD "\n\t"
                    "s_nop 2\n\t"
                    "s_lshr_b64 %[tmp], %[m], 1\n\t"
                    "s_and_b64 %[m], %[m], %[tmp]\n\t"
                    "s_or_b64 %[m], %[m], %[odd]\n\t"
                    "s_not_b64 %[m], %[m]\n\t"
                    "s_ff1_i32_b64 %[t], %[m]\n\t"
                    "s_lshr_b32 %[t], %[t], 1\n\t"
                    "s_min_u32 %[t], %[t], 32\n\t"
                    "s_cmp_le_u32 %[t], %[rdy]\n\t"
                    "s_cbranch_scc1 110f\n\t"
                    "s_mov_b32 %[rdy], %[t]\n\t"
                    "s_add_u32 %[t2], %[t], %[c64]\n\t"
                    "v_mov_b32 v112, %[t2]\n\t"
                    "global_store_dword %[mbx], v112, off sc1\n\t"
                    "s_sub_u32 %[t2], %[t], 3\n\t"
                    "s_cmp_ge_u32 %[t], 32\n\t"
                    "s_cselect_b32 %[t2], 1000, %[t2]\n\t"
                    "v_mov_b32 v111, %[t2]\n\t"
                    "ds_write_b32 %[lrdy], v111\n\t"
                    PTR_("s_memrealtime s[98:99]")
                    PTR_("s_min_u32 m0, %[t], 32")
                    PTR_("s_waitcnt lgkmcnt(0)")
                    PTR_("v_writelane_b32 %[tl], s98, m0")
                    PTR_("s_cmp_ge_u32 %[t], 32")
                    "s_cbranch_scc1 2f\n\t"
                    "110:\n\t"
                    "s_sleep " POLL_SLEEP_S "\n\t"
                    "s_waitcnt vmcnt(5)\n\t"
                    "v_subrev_u32 v110, %[call], v102\n\t"
                    "v_cmp_le_i32 %[m], 0, v110\n\t"
                    "global_load_dword v102, %[addr], off " POLL_MOD "\n\t"
                    "s_nop 2\n\t"
                    "s_lshr_b64 %[tmp], %[m], 1\n\t"
                    "s_and_b64 %[m], %[m], %[tmp]\n\t"
                    "s_or_b64 %[m], %[m], %[odd]\n\t"
                    "s_not_b64 %[m], %[m]\n\t"
                    "s_ff1_i32_b64 %[t], %[m]\n\t"
                    "s_lshr_b32 %[t], %[t], 1\n\t"
                    "s_min_u32 %[t], %[t], 32\n\t"
                    "s_cmp_le_u32 %[t], %[rdy]\n\t"
                    "s_cbranch_scc1 120f\n\t"
                    "s_mov_b32 %[rdy], %[t]\n\t"
                    "s_add_u32 %[t2], %[t], %[c64]\n\t"
                    "v_mov_b32 v112, %[t2]\n\t"
                    "global_store_dword %[mbx], v112, off sc1\n\t"
                    "s_sub_u32 %[t2], %[t], 3\n\t"
                    "s_cmp_ge_u32 %[t], 32\n\t"
                    "s_cselect_b32 %[t2], 1000, %[t2]\n\t"
                    "v_mov_b32 v111, %[t2]\n\t"
                    "ds_write_b32 %[lrdy], v111\n\t"
                    PTR_("s_memrealtime s[98:99]")
                    PTR_("s_min_u32 m0, %[t], 32")
                    PTR_("s_waitcnt lgkmcnt(0)")
                    PTR_("v_writelane_b32 %[tl], s98, m0")
                    PTR_("s_cmp_ge_u32 %[t], 32")
                    "s_cbranch_scc1 2f\n\t"
                    "120:\n\t"
                    "s_sleep " POLL_SLEEP_S "\n\t"
                    "s_waitcnt vmcnt(5)\n\t"
                    "v_subrev_u32 v110, %[call], v103\n\t"
                    "v_cmp_le_i32 %[m], 0, v110\n\t"
                    "global_load_dword v103, %[addr], off " POLL_MOD "\n\t"
                    "s_nop 2\n\t"
                    "s_lshr_b64 %[tmp], %[m], 1\n\t"
                    "s_and_b64 %[m], %[m], %[tmp]\n\t"
                    "s_or_b64 %[m], %[m], %[odd]\n\t"
                    "s_not_b64 %[m], %[m]\n\t"
                    "s_ff1_i32_b64 %[t], %[m]\n\t"
                    "s_lshr_b32 %[t], %[t], 1\n\t"
                    "s_min_u32 %[t], %[t], 32\n\t"
                    "s_cmp_le_u32 %[t], %[rdy]\n\t"
                    "s_cbranch_scc1 130f\n\t"
                    "s_mov_b32 %[rdy], %[t]\n\t"
                    "s_add_u32 %[t2], %[t], %[c64]\n\t"
                    "v_mov_b32 v112, %[t2]\n\t"
                    "global_store_dword %[mbx], v112, off sc1\n\t"
                    "s_sub_u32 %[t2], %[t], 3\n\t"
                    "s_cmp_ge_u32 %[t], 32\n\t"
                    "s_cselect_b32 %[t2], 1000, %[t2]\n\t"
                    "v_mov_b32 v111, %[t2]\n\t"
                    "ds_write_b32 %[lrdy], v111\n\t"
                    PTR_("s_memrealtime s[98:99]")
                    PTR_("s_min_u32 m0, %[t], 32")
                    PTR_("s_waitcnt lgkmcnt(0)")
                    PTR_("v_writelane_b32 %[tl], s98, m0")
                    PTR_("s_cmp_ge_u32 %[t], 32")
                    "s_cbranch_scc1 2f\n\t"
                    "130:\n\t"
                    "s_sleep " POLL_SLEEP_S "\n\t"
                    "s_waitcnt vmcnt(5)\n\t"
                    "v_subrev_u32 v110, %[call], v104\n\t"
                    "v_cmp_le_i32 %[m], 0, v110\n\t"
                    "global_load_dword v104, %[addr], off " POLL_MOD "\n\t"
                    "s_nop 2\n\t"
                    "s_lshr_b64 %[tmp], %[m], 1\n\t"
                    "s_and_b64 %[m], %[m], %[tmp]\n\t"
                    "s_or_b64 %[m], %[m], %[odd]\n\t"
                    "s_not_b64 %[m], %[m]\n\t"
                    "s_ff1_i32_b64 %[t], %[m]\n\t"
                    "s_lshr_b32 %[t], %[t], 1\n\t"
                    "s_min_u32 %[t], %[t], 32\n\t"
                    "s_cmp_le_u32 %[t], %[rdy]\n\t"
                    "s_cbranch_scc1 140f\n\t"
                    "s_mov_b32 %[rdy], %[t]\n\t"
                    "s_add_u32 %[t2], %[t], %[c64]\n\t"
                    "v_mov_b32 v112, %[t2]\n\t"
                    "global_store_dword %[mbx], v112, off sc1\n\t"
                    "s_sub_u32 %[t2], %[t], 3\n\t"
                    "s_cmp_ge_u32 %[t], 32\n\t"
                    "s_cselect_b32 %[t2], 1000, %[t2]\n\t"
                    "v_mov_b32 v111, %[t2]\n\t"
                    "ds_write_b32 %[lrdy], v111\n\t"
                    PTR_("s_memrealtime s[98:99]")
                    PTR_("s_min_u32 m0, %[t], 32")
                    PTR_("s_waitcnt lgkmcnt(0)")
                    PTR_("v_writelane_b32 %[tl], s98, m0")
                    PTR_("s_cmp_ge_u32 %[t], 32")
                    "s_cbranch_scc1 2f\n\t"
                    "140:\n\t"
                    "s_sleep " POLL_SLEEP_S "\n\t"
                    "s_waitcnt vmcnt(5)\n\t"
                    "v_subrev_u32 v110, %[call], v105\n\t"
                    "v_cmp_le_i32 %[m], 0, v110\n\t"
                    "global_load_dword v105, %[addr], off " POLL_MOD "\n\t"
                    "s_nop 2\n\t"
                    "s_lshr_b64 %[tmp], %[m], 1\n\t"
                    "s_and_b64 %[m], %[m], %[tmp]\n\t"
                    "s_or_b64 %[m], %[m], %[odd]\n\t"
                    "s_not_b64 %[m], %[m]\n\t"
                    "s_ff1_i32_b64 %[t], %[m]\n\t"
                    "s_lshr_b32 %[t], %[t], 1\n\t"
                    "s_min_u32 %[t], %[t], 32\n\t"
                    "s_cmp_le_u32 %[t], %[rdy]\n\t"
                    "s_cbranch_scc1 150f\n\t"
                    "s_mov_b32 %[rdy], %[t]\n\t"
                    "s_add_u32 %[t2], %[t], %[c64]\n\t"
                    "v_mov_b32 v112, %[t2]\n\t"
                    "global_store_dword %[mbx], v112, off sc1\n\t"
                    "s_sub_u32 %[t2], %[t], 3\n\t"
                    "s_cmp_ge_u32 %[t], 32\n\t"
                    "s_cselect_b32 %[t2], 1000, %[t2]\n\t"
                    "v_mov_b32 v111, %[t2]\n\t"
                    "ds_write_b32 %[lrdy], v111\n\t"
                    PTR_("s_memrealtime s[98:99]")
                    PTR_("s_min_u32 m0, %[t], 32")
                    PTR_("s_waitcnt lgkmcnt(0)")
                    PTR_("v_writelane_b32 %[tl], s98, m0")
                    PTR_("s_cmp_ge_u32 %[t], 32")
                    "s_cbranch_scc1 2f\n\t"
                    "150:\n\t"
                    "s_sleep " POLL_SLEEP_S "\n\t"
                    "s_sub_u32 %[it], %[it], 1\n\t"
                    "s_cmp_lg_u32 %[it], 0\n\t"
                    "s_cbranch_scc1 1b\n\t"
                    "2:\n\t"
                    "s_waitcnt vmcnt(0)\n\t"
                    : [m] "=&s"(m), [tmp] "=&s"(tmp), [it] "+s"(iters), [t] "=&s"(t), [t2] "=&s"(t2), [rdy] "+s"(rdy)
                    : [addr] "v"(fl), [call] "s"(call), [odd] "s"(0xAAAAAAAAAAAAAAAAull), [lrdy] "v"(L_READY), [mbx] "v"(mbx), [c64] "s"(call * 64u)
                    : "memory", "scc", "s98", "s99", "v100", "v101", "v102", "v103", "v104", "v105", "v110", "v111", "v112");
            } else {
                asm volatile(
                    "global_load_dword v100, %[addr], off " POLL_MOD "\n\t"
                    "s_sleep " POLL_SLEEP_S "\n\t"
                    "global_load_dword v101, %[addr], off " POLL_MOD "\n\t"
                    "s_sleep " POLL_SLEEP_S "\n\t"
                    "global_load_dword v102, %[addr], off " POLL_MOD "\n\t"
                    "s_sleep " POLL_SLEEP_S "\n\t"
                    "global_load_dword v103, %[addr], off " POLL_MOD "\n\t"
                    "s_sleep " POLL_SLEEP_S "\n\t"
                    "1:\n\t"
                    "s_waitcnt vmcnt(3)\n\t"
                    "v_readfirstlane_b32 %[t], v100\n\t"
                    "global_load_dword v100, %[addr], off " POLL_MOD "\n\t"
                    "s_sub_i32 %[t], %[t], %[call]\n\t"
                    "s_max_i32 %[t], %[t], 0\n\t"
                    "s_cmp_le_u32 %[t], %[rdy]\n\t"
                    "s_cbranch_scc1 100f\n\t"
                    "s_mov_b32 %[rdy], %[t]\n\t"
                    "s_sub_u32 %[t2], %[t], 3\n\t"
                    "s_cmp_ge_u32 %[t], 32\n\t"
                    "s_cselect_b32 %[t2], 1000, %[t2]\n\t"
                    "v_mov_b32 v111, %[t2]\n\t"
                    "ds_write_b32 %[lrdy], v111\n\t"
                    PTR_("s_memrealtime s[98:99]")
                    PTR_("s_min_u32 m0, %[t], 32")
                    PTR_("s_waitcnt lgkmcnt(0)")
                    PTR_("v_writelane_b32 %[tl], s98, m0")
                    PTR_("s_cmp_ge_u32 %[t], 32")
                    "s_cbranch_scc1 2f\n\t"
                    "100:\n\t"
                    "s_sleep " POLL_SLEEP_S "\n\t"
                    "s_waitcnt vmcnt(3)\n\t"
                    "v_readfirstlane_b32 %[t], v101\n\t"
                    "global_load_dword v101, %[addr], off " POLL_MOD "\n\t"
                    "s_sub_i32 %[t], %[t], %[call]\n\t"
                    "s_max_i32 %[t], %[t], 0\n\t"
                    "s_cmp_le_u32 %[t], %[rdy]\n\t"
                    "s_cbranch_scc1 110f\n\t"
                    "s_mov_b32 %[rdy], %[t]\n\t"
                    "s_sub_u32 %[t2], %[t], 3\n\t"
                    "s_cmp_ge_u32 %[t], 32\n\t"
                    "s_cselect_b32 %[t2], 1000, %[t2]\n\t"
                    "v_mov_b32 v111, %[t2]\n\t"
                    "ds_write_b32 %[lrdy], v111\n\t"
                    PTR_("s_memrealtime s[98:99]")
                    PTR_("s_min_u32 m0, %[t], 32")
                    PTR_("s_waitcnt lgkmcnt(0)")
                    PTR_("v_writelane_b32 %[tl], s98, m0")
                    PTR_("s_cmp_ge_u32 %[t], 32")
                    "s_cbranch_scc1 2f\n\t"
                    "110:\n\t"
                    "s_sleep " POLL_SLEEP_S "\n\t"
                    "s_waitcnt vmcnt(3)\n\t"
                    "v_readfirstlane_b32 %[t], v102\n\t"
                    "global_load_dword v102, %[addr], off " POLL_MOD "\n\t"
                    "s_sub_i32 %[t], %[t], %[call]\n\t"
                    "s_max_i32 %[t], %[t], 0\n\t"
                    "s_cmp_le_u32 %[t], %[rdy]\n\t"
                    "s_cbranch_scc1 120f\n\t"
                    "s_mov_b32 %[rdy], %[t]\n\t"
                    "s_sub_u32 %[t2], %[t], 3\n\t"
                    "s_cmp_ge_u32 %[t], 32\n\t"
                    "s_cselect_b32 %[t2], 1000, %[t2]\n\t"
                    "v_mov_b32 v111, %[t2]\n\t"
                    "ds_write_b32 %[lrdy], v111\n\t"
                    PTR_("s_memrealtime s[98:99]")
                    PTR_("s_min_u32 m0, %[t], 32")
                    PTR_("s_waitcnt lgkmcnt(0)")
                    PTR_("v_writelane_b32 %[tl], s98, m0")
                    PTR_("s_cmp_ge_u32 %[t], 32")
                    "s_cbranch_scc1 2f\n\t"
                    "120:\n\t"
                    "s_sleep " POLL_SLEEP_S "\n\t"
                    "s_waitcnt vmcnt(3)\n\t"
                    "v_readfirstlane_b32 %[t], v103\n\t"
                    "global_load_dword v103, %[addr], off " POLL_MOD "\n\t"
                    "s_sub_i32 %[t], %[t], %[call]\n\t"
                    "s_max_i32 %[t], %[t], 0\n\t"
                    "s_cmp_le_u32 %[t], %[rdy]\n\t"
                    "s_cbranch_scc1 130f\n\t"
                    "s_mov_b32 %[rdy], %[t]\n\t"
                    "s_sub_u32 %[t2], %[t], 3\n\t"
                    "s_cmp_ge_u32 %[t], 32\n\t"
                    "s_cselect_b32 %[t2], 1000, %[t2]\n\t"
                    "v_mov_b32 v111, %[t2]\n\t"
                    "ds_write_b32 %[lrdy], v111\n\t"
                    PTR_("s_memrealtime s[98:99]")
                    PTR_("s_min_u32 m0, %[t], 32")
                    PTR_("s_waitcnt lgkmcnt(0)")
                    PTR_("v_writelane_b32 %[tl], s98, m0")
                    PTR_("s_cmp_ge_u32 %[t], 32")
                    "s_cbranch_scc1 2f\n\t"
                    "130:\n\t"
                    "s_sleep " POLL_SLEEP_S "\n\t"
                    "s_sub_u32 %[it], %[it], 1\n\t"
                    "s_cmp_lg_u32 %[it], 0\n\t"
                    "s_cbranch_scc1 1b\n\t"
                    "2:\n\t"
                    "s_waitcnt vmcnt(0)\n\t"
                    : [it] "+s"(iters), [t] "=&s"(t), [t2] "=&s"(t2), [rdy] "+s"(rdy)
                    : [addr] "v"(fl), [call] "s"(call * 64u), [lrdy] "v"(L_READY)
                    : "memory", "scc", "s98", "s99", "v100", "v101", "v102", "v103", "v110", "v111", "v112");
            }
            ready = rfl(rdy);
            if (ready < NSTAGE) {
                unsigned tv = (unsigned)rfl((int)__hip_atomic_load(tmo, __ATOMIC_RELAXED, __HIP_MEMORY_SCOPE_AGENT));
                if (tv == call || rt_clock() - t_start > DEADLINE_TICKS) {
                    if (lane == 0) { __hip_atomic_store(tmo, call, __ATOMIC_RELAXED, __HIP_MEMORY_SCOPE_AGENT); lds_put(L_BAD, 1); lds_put(L_READY, 1000); }
                    ready = NSTAGE;
                }
            }
        }
        asm volatile("s_waitcnt vmcnt(0)" ::: "memory");
        if (lane == 0) __hip_atomic_store(ctr, call, __ATOMIC_RELAXED, __HIP_MEMORY_SCOPE_AGENT);
        return;
    }

    if (wave == 3) {
        __builtin_amdgcn_s_barrier();
        const int wi = bid >> 3;
#pragma unroll
        for (int i = WARM_FIRST; i < 8; ++i) {
            while ((long long)(rt_clock() - t_start) < WARM_DELAY + (i - WARM_FIRST) * WARM_STEP) __builtin_amdgcn_s_sleep(2);
            int row;
            if (i == 0) row = rank * KC + wi * 16;
            else {
                const int u = (i - 1) * 32 + wi, kq = u / 56, r = u - kq * 56;
                row = ((rank + 1 + (r >> 3)) & 7) * KC + kq * 128 + (r & 7) * 16;
            }
            asm volatile("global_load_dword v121, %0, off" :: "v"(P.w + (size_t)(row + (lane >> 2)) * NN + xcd * 512 + (lane & 3) * 128) : "memory", "v121");
        }
        WAITV(0);
        return;
    }

    if (wave >= 8) {
        const int lw = wave - 8;
        const unsigned w_voff = (unsigned)((lane >> 3) * NN + 16 * ((lane & 7) ^ (((lane >> 4) & 3) << 1)));
        const char* const wbase = (const char*)P.w + nb * 128;
        __builtin_amdgcn_s_barrier();
        if (bid < ADJ) { while (min(lds_get(L_LANDB + 56), lds_get(L_LANDB + 60)) < 1) __builtin_amdgcn_s_sleep(2); }
        int cons = 0;
        int pend = -1;
        bool comm_done = false;
#pragma unroll 1
        for (int s = lw; s < NSTAGE; s += NLW) {
            int p, kq; stage_info(s, rank, mb, p, kq);
            const char* sb = wbase + (size_t)(p * KC + kq * 128) * NN;
            const unsigned dst = lds0 + LDS_B + (unsigned)((s & (NB - 1)) * 16384);
            if (s - NB >= cons) cons = lds_min4(L_CONS);
            if (s - NB >= cons || !W2) {
                if (pend >= 0) { WAITV(0); if (lane == 0) lds_put(L_LANDB + (unsigned)(pend & 3) * 4u, pend + 1); pend = -1; }
                while (s - NB >= cons) { cons = lds_min4(L_CONS); if (s - NB >= cons) __builtin_amdgcn_s_sleep(LSLEEP); }
            }
#pragma unroll
            for (int i = 0; i < 16; ++i) {
                unsigned keep;
                asm volatile("s_mov_b32 %0, m0\n\ts_mov_b32 m0, %3\n\ts_nop 0\n\t"
                             "global_load_lds_dwordx4 %1, %2\n\ts_mov_b32 m0, %0"
                             : "=&s"(keep) : "v"(w_voff), "s"(sb + (size_t)i * 8 * NN), "s"(dst + i * 1024) : "memory");
            }
            if (pend >= 0) { WAITV(16); if (lane == 0) lds_put(L_LANDB + (unsigned)(pend & 3) * 4u, pend + 1); }
            pend = s;
        }
        WAITV(0);
        if (lane == 0) lds_put(L_LANDB + (unsigned)lw * 4u, 1000);
        return;
    }

    const int cw = wave - 4;
    const int mh = cw & 1, kh = cw >> 1;
    const int g = lane >> 4, q = (lane & 15) >> 1, pp = lane & 1;
    const unsigned trow = lds0 + LDS_B + (unsigned)(kh * 8192 + (16 * (g >> 1) + q) * 128 + (g & 1) * 16 + pp * 8);
    const int sx = ((q >> 1) & 3) ^ (2 * kh);
    const unsigned tb0 = trow + (unsigned)((0 ^ sx) * 32), tb1 = trow + (unsigned)((1 ^ sx) * 32);
    const unsigned tb2 = trow + (unsigned)((2 ^ sx) * 32), tb3 = trow + (unsigned)((3 ^ sx) * 32);
    unsigned tab;
    { int p, kq; stage_info(lane < NSTAGE ? lane : NSTAGE - 1, rank, mb, p, kq); tab = (unsigned)(kq * 8 + p) * 65536u; }
    const unsigned voff = (unsigned)(((mb * 2 + mh) * 4 + 2 * kh) * 1024 + lane * 16);
    const char* const abase = myws + STG_OFF + (size_t)(call & (NSTG - 1)) * STG_BYTES;

    const float scale0 = *(const __attribute__((address_space(4))) float*)(uintptr_t)P.sx * *(const __attribute__((address_space(4))) float*)(uintptr_t)P.sw;
    v16i c0 = {}, c1 = {}, c2 = {}, c3 = {};
    __builtin_amdgcn_s_barrier();
    asm volatile("" ::: "memory");
    while (lds_get(L_READY) < 1) __builtin_amdgcn_s_sleep(1);
    while (lds_get(L_LANDB) < 1) __builtin_amdgcn_s_sleep(1);
    __builtin_amdgcn_s_setprio(2);
    {
        int sthr, st;
#define CNT_(x) ""
#define RD_(x) x "\n\t"
#define AL_(x) x "\n\t"
#define MF_(x) x "\n\t"
        asm volatile(
        "v_mov_b32 v152, %[tb0]\n\t"
        "v_mov_b32 v153, %[tb1]\n\t"
        "v_mov_b32 v154, %[tb2]\n\t"
        "v_mov_b32 v155, %[tb3]\n\t"
        "v_add_u32 v156, 0x10000, v152\n\t"
        "v_add_u32 v157, 0x10000, v153\n\t"
        "v_add_u32 v158, 0x10000, v154\n\t"
        "v_add_u32 v159, 0x10000, v155\n\t"
        "v_mov_b32 v148, 0\n\t"
        "v_mov_b32 v149, 0\n\t"
        "v_mov_b32 v151, %[landb]\n\t"
        "v_mov_b32 v161, %[tab]\n\t"
        "s_mov_b32 %[thr], 1\n\t"
        "v_readlane_b32 %[t], v161, 0\n\t"
        "s_nop 1\n\t"
        "v_add_u32 v160, %[t], %[voff]\n\t"
        AL_("global_load_dwordx4 v[84:87], v160, %[abase]")
        AL_("global_load_dwordx4 v[88:91], v160, %[abase] offset:1024")
        "v_readlane_b32 %[t], v161, 1\n\t"
        "s_nop 1\n\t"
        "v_add_u32 v160, %[t], %[voff]\n\t"
        AL_("global_load_dwordx4 v[92:95], v160, %[abase]")
        AL_("global_load_dwordx4 v[96:99], v160, %[abase] offset:1024")
        "v_readlane_b32 %[t], v161, 2\n\t"
        "s_nop 1\n\t"
        "v_add_u32 v160, %[t], %[voff]\n\t"
        AL_("global_load_dwordx4 v[100:103], v160, %[abase]")
        AL_("global_load_dwordx4 v[104:107], v160, %[abase] offset:1024")
        "v_readlane_b32 %[t], v161, 3\n\t"
        "s_nop 1\n\t"
        "v_add_u32 v160, %[t], %[voff]\n\t"
        AL_("global_load_dwordx4 v[108:111], v160, %[abase]")
        AL_("global_load_dwordx4 v[112:115], v160, %[abase] offset:1024")
        RD_("ds_read_b64_tr_b8 v[116:117], v152 offset:0")
        RD_("ds_read_b64_tr_b8 v[118:119], v152 offset:1024")
        RD_("ds_read_b64_tr_b8 v[120:121], v153 offset:0")
        RD_("ds_read_b64_tr_b8 v[122:123], v153 offset:1024")
        RD_("ds_read_b64_tr_b8 v[124:125], v154 offset:0")
        RD_("ds_read_b64_tr_b8 v[126:127], v154 offset:1024")
        RD_("ds_read_b64_tr_b8 v[128:129], v155 offset:0")
        RD_("ds_read_b64_tr_b8 v[130:131], v155 offset:1024")
        RD_("ds_read_b64_tr_b8 v[132:133], v152 offset:4096")
        RD_("ds_read_b64_tr_b8 v[134:135], v152 offset:5120")
        RD_("ds_read_b64_tr_b8 v[136:137], v153 offset:4096")
        RD_("ds_read_b64_tr_b8 v[138:139], v153 offset:5120")
        RD_("ds_read_b64_tr_b8 v[140:141], v154 offset:4096")
        RD_("ds_read_b64_tr_b8 v[142:143], v154 offset:5120")
        RD_("ds_read_b64_tr_b8 v[144:145], v155 offset:4096")
        RD_("ds_read_b64_tr_b8 v[146:147], v155 offset:5120")
        "s_waitcnt lgkmcnt(0)\n\t"
        "1:\n\t"
        "s_add_u32 %[thr], %[thr], 1\n\t"
        "s_waitcnt lgkmcnt(8) vmcnt(7)\n\t"
        MF_("v_mfma_i32_32x32x32_i8 %[c0], v[84:87], v[116:119], %[c0]")
        "v_min_i32 v150, v148, v149\n\t"
        "v_readlane_b32 %[t], v161, 4\n\t"
        "v_cmp_gt_i32 vcc, %[thr], v150\n\t"
        MF_("v_mfma_i32_32x32x32_i8 %[c1], v[84:87], v[120:123], %[c1]")
        "v_add_u32 v160, %[t], %[voff]\n\t"
        "s_cbranch_vccnz 10f\n\t"
        "11:\n\t"
        RD_("ds_read_b64_tr_b8 v[116:117], v152 offset:16384")
        RD_("ds_read_b64_tr_b8 v[118:119], v152 offset:17408")
        RD_("ds_read_b64_tr_b8 v[120:121], v153 offset:16384")
        RD_("ds_read_b64_tr_b8 v[122:123], v153 offset:17408")
        MF_("v_mfma_i32_32x32x32_i8 %[c2], v[84:87], v[124:127], %[c2]")
        RD_("ds_read_b64_tr_b8 v[124:125], v154 offset:16384")
        RD_("ds_read_b64_tr_b8 v[126:127], v154 offset:17408")
        MF_("v_mfma_i32_32x32x32_i8 %[c3], v[84:87], v[128:131], %[c3]")
        RD_("ds_read_b64_tr_b8 v[128:129], v155 offset:16384")
        RD_("ds_read_b64_tr_b8 v[130:131], v155 offset:17408")
        AL_("global_load_dwordx4 v[84:87], v160, %[abase]")
        "s_waitcnt lgkmcnt(8) vmcnt(7)\n\t"
        MF_("v_mfma_i32_32x32x32_i8 %[c0], v[88:91], v[132:135], %[c0]")
        "ds_read2_b32 v[148:149], v151 offset0:2 offset1:12\n\t"
        "v_add_u32 v150, -1, %[thr]\n\t"
        "ds_write_b32 %[cons], v150\n\t"
        RD_("ds_read_b64_tr_b8 v[132:133], v152 offset:20480")
        RD_("ds_read_b64_tr_b8 v[134:135], v152 offset:21504")
        MF_("v_mfma_i32_32x32x32_i8 %[c1], v[88:91], v[136:139], %[c1]")
        RD_("ds_read_b64_tr_b8 v[136:137], v153 offset:20480")
        RD_("ds_read_b64_tr_b8 v[138:139], v153 offset:21504")
        MF_("v_mfma_i32_32x32x32_i8 %[c2], v[88:91], v[140:143], %[c2]")
        RD_("ds_read_b64_tr_b8 v[140:141], v154 offset:20480")
        RD_("ds_read_b64_tr_b8 v[142:143], v154 offset:21504")
        MF_("v_mfma_i32_32x32x32_i8 %[c3], v[88:91], v[144:147], %[c3]")
        RD_("ds_read_b64_tr_b8 v[144:145], v155 offset:20480")
        RD_("ds_read_b64_tr_b8 v[146:147], v155 offset:21504")
        AL_("global_load_dwordx4 v[88:91], v160, %[abase] offset:1024")
        "s_add_u32 %[thr], %[thr], 1\n\t"
        "s_waitcnt lgkmcnt(8) vmcnt(7)\n\t"
        MF_("v_mfma_i32_32x32x32_i8 %[c0], v[92:95], v[116:119], %[c0]")
        "v_min_i32 v150, v148, v149\n\t"
        "v_readlane_b32 %[t], v161, 5\n\t"
        "v_cmp_gt_i32 vcc, %[thr], v150\n\t"
        MF_("v_mfma_i32_32x32x32_i8 %[c1], v[92:95], v[120:123], %[c1]")
        "v_add_u32 v160, %[t], %[voff]\n\t"
        "s_cbranch_vccnz 20f\n\t"
        "21:\n\t"
        RD_("ds_read_b64_tr_b8 v[116:117], v152 offset:32768")
        RD_("ds_read_b64_tr_b8 v[118:119], v152 offset:33792")
        RD_("ds_read_b64_tr_b8 v[120:121], v153 offset:32768")
        RD_("ds_read_b64_tr_b8 v[122:123], v153 offset:33792")
        MF_("v_mfma_i32_32x32x32_i8 %[c2], v[92:95], v[124:127], %[c2]")
        RD_("ds_read_b64_tr_b8 v[124:125], v154 offset:32768")
        RD_("ds_read_b64_tr_b8 v[126:127], v154 offset:33792")
        MF_("v_mfma_i32_32x32x32_i8 %[c3], v[92:95], v[128:131], %[c3]")
        RD_("ds_read_b64_tr_b8 v[128:129], v155 offset:32768")
        RD_("ds_read_b64_tr_b8 v[130:131], v155 offset:33792")
        AL_("global_load_dwordx4 v[92:95], v160, %[abase]")
        "s_waitcnt lgkmcnt(8) vmcnt(7)\n\t"
        MF_("v_mfma_i32_32x32x32_i8 %[c0], v[96:99], v[132:135], %[c0]")
        "ds_read2_b32 v[148:149], v151 offset0:3 offset1:12\n\t"
        "v_add_u32 v150, -1, %[thr]\n\t"
        "ds_write_b32 %[cons], v150\n\t"
        RD_("ds_read_b64_tr_b8 v[132:133], v152 offset:36864")
        RD_("ds_read_b64_tr_b8 v[134:135], v152 offset:37888")
        MF_("v_mfma_i32_32x32x32_i8 %[c1], v[96:99], v[136:139], %[c1]")
        RD_("ds_read_b64_tr_b8 v[136:137], v153 offset:36864")
        RD_("ds_read_b64_tr_b8 v[138:139], v153 offset:37888")
        MF_("v_mfma_i32_32x32x32_i8 %[c2], v[96:99], v[140:143], %[c2]")
        RD_("ds_read_b64_tr_b8 v[140:141], v154 offset:36864")
        RD_("ds_read_b64_tr_b8 v[142:143], v154 offset:37888")
        MF_("v_mfma_i32_32x32x32_i8 %[c3], v[96:99], v[144:147], %[c3]")
        RD_("ds_read_b64_tr_b8 v[144:145], v155 offset:36864")
        RD_("ds_read_b64_tr_b8 v[146:147], v155 offset:37888")
        AL_("global_load_dwordx4 v[96:99], v160, %[abase] offset:1024")
        "s_add_u32 %[thr], %[thr], 1\n\t"
        "s_waitcnt lgkmcnt(8) vmcnt(7)\n\t"
        MF_("v_mfma_i32_32x32x32_i8 %[c0], v[100:103], v[116:119], %[c0]")
        "v_min_i32 v150, v148, v149\n\t"
        "v_readlane_b32 %[t], v161, 6\n\t"
        "v_cmp_gt_i32 vcc, %[thr], v150\n\t"
        MF_("v_mfma_i32_32x32x32_i8 %[c1], v[100:103], v[120:123], %[c1]")
        "v_add_u32 v160, %[t], %[voff]\n\t"
        "s_cbranch_vccnz 30f\n\t"
        "31:\n\t"
        RD_("ds_read_b64_tr_b8 v[116:117], v152 offset:49152")
        RD_("ds_read_b64_tr_b8 v[118:119], v152 offset:50176")
        RD_("ds_read_b64_tr_b8 v[120:121], v153 offset:49152")
        RD_("ds_read_b64_tr_b8 v[122:123], v153 offset:50176")
        MF_("v_mfma_i32_32x32x32_i8 %[c2], v[100:103], v[124:127], %[c2]")
        RD_("ds_read_b64_tr_b8 v[124:125], v154 offset:49152")
        RD_("ds_read_b64_tr_b8 v[126:127], v154 offset:50176")
        MF_("v_mfma_i32_32x32x32_i8 %[c3], v[100:103], v[128:131], %[c3]")
        RD_("ds_read_b64_tr_b8 v[128:129], v155 offset:49152")
        RD_("ds_read_b64_tr_b8 v[130:131], v155 offset:50176")
        AL_("global_load_dwordx4 v[100:103], v160, %[abase]")
        "s_waitcnt lgkmcnt(8) vmcnt(7)\n\t"
        MF_("v_mfma_i32_32x32x32_i8 %[c0], v[104:107], v[132:135], %[c0]")
        "ds_read2_b32 v[148:149], v151 offset0:0 offset1:12\n\t"
        "v_add_u32 v150, -1, %[thr]\n\t"
        "ds_write_b32 %[cons], v150\n\t"
        RD_("ds_read_b64_tr_b8 v[132:133], v152 offset:53248")
        RD_("ds_read_b64_tr_b8 v[134:135], v152 offset:54272")
        MF_("v_mfma_i32_32x32x32_i8 %[c1], v[104:107], v[136:139], %[c1]")
        RD_("ds_read_b64_tr_b8 v[136:137], v153 offset:53248")
        RD_("ds_read_b64_tr_b8 v[138:139], v153 offset:54272")
        MF_("v_mfma_i32_32x32x32_i8 %[c2], v[104:107], v[140:143], %[c2]")
        RD_("ds_read_b64_tr_b8 v[140:141], v154 offset:53248")
        RD_("ds_read_b64_tr_b8 v[142:143], v154 offset:54272")
        MF_("v_mfma_i32_32x32x32_i8 %[c3], v[104:107], v[144:147], %[c3]")
        RD_("ds_read_b64_tr_b8 v[144:145], v155 offset:53248")
        RD_("ds_read_b64_tr_b8 v[146:147], v155 offset:54272")
        AL_("global_load_dwordx4 v[104:107], v160, %[abase] offset:1024")
        "s_add_u32 %[thr], %[thr], 1\n\t"
        "s_waitcnt lgkmcnt(8) vmcnt(7)\n\t"
        MF_("v_mfma_i32_32x32x32_i8 %[c0], v[108:111], v[116:119], %[c0]")
        "v_min_i32 v150, v148, v149\n\t"
        "v_readlane_b32 %[t], v161, 7\n\t"
        "v_cmp_gt_i32 vcc, %[thr], v150\n\t"
        MF_("v_mfma_i32_32x32x32_i8 %[c1], v[108:111], v[120:123], %[c1]")
        "v_add_u32 v160, %[t], %[voff]\n\t"
        "s_cbranch_vccnz 40f\n\t"
        "41:\n\t"
        RD_("ds_read_b64_tr_b8 v[116:117], v156 offset:0")
        RD_("ds_read_b64_tr_b8 v[118:119], v156 offset:1024")
        RD_("ds_read_b64_tr_b8 v[120:121], v157 offset:0")
        RD_("ds_read_b64_tr_b8 v[122:123], v157 offset:1024")
        MF_("v_mfma_i32_32x32x32_i8 %[c2], v[108:111], v[124:127], %[c2]")
        RD_("ds_read_b64_tr_b8 v[124:125], v158 offset:0")
        RD_("ds_read_b64_tr_b8 v[126:127], v158 offset:1024")
        MF_("v_mfma_i32_32x32x32_i8 %[c3], v[108:111], v[128:131], %[c3]")
        RD_("ds_read_b64_tr_b8 v[128:129], v159 offset:0")
        RD_("ds_read_b64_tr_b8 v[130:131], v159 offset:1024")
        AL_("global_load_dwordx4 v[108:111], v160, %[abase]")
        "s_waitcnt lgkmcnt(8) vmcnt(7)\n\t"
        MF_("v_mfma_i32_32x32x32_i8 %[c0], v[112:115], v[132:135], %[c0]")
        "ds_read2_b32 v[148:149], v151 offset0:1 offset1:12\n\t"
        "v_add_u32 v150, -1, %[thr]\n\t"
        "ds_write_b32 %[cons], v150\n\t"
        RD_("ds_read_b64_tr_b8 v[132:133], v156 offset:4096")
        RD_("ds_read_b64_tr_b8 v[134:135], v156 offset:5120")
        MF_("v_mfma_i32_32x32x32_i8 %[c1], v[112:115], v[136:139], %[c1]")
        RD_("ds_read_b64_tr_b8 v[136:137], v157 offset:4096")
        RD_("ds_read_b64_tr_b8 v[138:139], v157 offset:5120")
        MF_("v_mfma_i32_32x32x32_i8 %[c2], v[112:115], v[140:143], %[c2]")
        RD_("ds_read_b64_tr_b8 v[140:141], v158 offset:4096")
        RD_("ds_read_b64_tr_b8 v[142:143], v158 offset:5120")
        MF_("v_mfma_i32_32x32x32_i8 %[c3], v[112:115], v[144:147], %[c3]")
        RD_("ds_read_b64_tr_b8 v[144:145], v159 offset:4096")
        RD_("ds_read_b64_tr_b8 v[146:147], v159 offset:5120")
        AL_("global_load_dwordx4 v[112:115], v160, %[abase] offset:1024")
        "s_add_u32 %[thr], %[thr], 1\n\t"
        "s_waitcnt lgkmcnt(8) vmcnt(7)\n\t"
        MF_("v_mfma_i32_32x32x32_i8 %[c0], v[84:87], v[116:119], %[c0]")
        "v_min_i32 v150, v148, v149\n\t"
        "v_readlane_b32 %[t], v161, 8\n\t"
        "v_cmp_gt_i32 vcc, %[thr], v150\n\t"
        MF_("v_mfma_i32_32x32x32_i8 %[c1], v[84:87], v[120:123], %[c1]")
        "v_add_u32 v160, %[t], %[voff]\n\t"
        "s_cbranch_vccnz 50f\n\t"
        "51:\n\t"
        RD_("ds_read_b64_tr_b8 v[116:117], v156 offset:16384")
        RD_("ds_read_b64_tr_b8 v[118:119], v156 offset:17408")
        RD_("ds_read_b64_tr_b8 v[120:121], v157 offset:16384")
        RD_("ds_read_b64_tr_b8 v[122:123], v157 offset:17408")
        MF_("v_mfma_i32_32x32x32_i8 %[c2], v[84:87], v[124:127], %[c2]")
        RD_("ds_read_b64_tr_b8 v[124:125], v158 offset:16384")
        RD_("ds_read_b64_tr_b8 v[126:127], v158 offset:17408")
        MF_("v_mfma_i32_32x32x32_i8 %[c3], v[84:87], v[128:131], %[c3]")
        RD_("ds_read_b64_tr_b8 v[128:129], v159 offset:16384")
        RD_("ds_read_b64_tr_b8 v[130:131], v159 offset:17408")
        AL_("global_load_dwordx4 v[84:87], v160, %[abase]")
        "s_waitcnt lgkmcnt(8) vmcnt(7)\n\t"
        MF_("v_mfma_i32_32x32x32_i8 %[c0], v[88:91], v[132:135], %[c0]")
        "ds_read2_b32 v[148:149], v151 offset0:2 offset1:12\n\t"
        "v_add_u32 v150, -1, %[thr]\n\t"
        "ds_write_b32 %[cons], v150\n\t"
        RD_("ds_read_b64_tr_b8 v[132:133], v156 offset:20480")
        RD_("ds_read_b64_tr_b8 v[134:135], v156 offset:21504")
        MF_("v_mfma_i32_32x32x32_i8 %[c1], v[88:91], v[136:139], %[c1]")
        RD_("ds_read_b64_tr_b8 v[136:137], v157 offset:20480")
        RD_("ds_read_b64_tr_b8 v[138:139], v157 offset:21504")
        MF_("v_mfma_i32_32x32x32_i8 %[c2], v[88:91], v[140:143], %[c2]")
        RD_("ds_read_b64_tr_b8 v[140:141], v158 offset:20480")
        RD_("ds_read_b64_tr_b8 v[142:143], v158 offset:21504")
        MF_("v_mfma_i32_32x32x32_i8 %[c3], v[88:91], v[144:147], %[c3]")
        RD_("ds_read_b64_tr_b8 v[144:145], v159 offset:20480")
        RD_("ds_read_b64_tr_b8 v[146:147], v159 offset:21504")
        AL_("global_load_dwordx4 v[88:91], v160, %[abase] offset:1024")
        "s_add_u32 %[thr], %[thr], 1\n\t"
        "s_waitcnt lgkmcnt(8) vmcnt(7)\n\t"
        MF_("v_mfma_i32_32x32x32_i8 %[c0], v[92:95], v[116:119], %[c0]")
        "v_min_i32 v150, v148, v149\n\t"
        "v_readlane_b32 %[t], v161, 9\n\t"
        "v_cmp_gt_i32 vcc, %[thr], v150\n\t"
        MF_("v_mfma_i32_32x32x32_i8 %[c1], v[92:95], v[120:123], %[c1]")
        "v_add_u32 v160, %[t], %[voff]\n\t"
        "s_cbranch_vccnz 60f\n\t"
        "61:\n\t"
        RD_("ds_read_b64_tr_b8 v[116:117], v156 offset:32768")
        RD_("ds_read_b64_tr_b8 v[118:119], v156 offset:33792")
        RD_("ds_read_b64_tr_b8 v[120:121], v157 offset:32768")
        RD_("ds_read_b64_tr_b8 v[122:123], v157 offset:33792")
        MF_("v_mfma_i32_32x32x32_i8 %[c2], v[92:95], v[124:127], %[c2]")
        RD_("ds_read_b64_tr_b8 v[124:125], v158 offset:32768")
        RD_("ds_read_b64_tr_b8 v[126:127], v158 offset:33792")
        MF_("v_mfma_i32_32x32x32_i8 %[c3], v[92:95], v[128:131], %[c3]")
        RD_("ds_read_b64_tr_b8 v[128:129], v159 offset:32768")
        RD_("ds_read_b64_tr_b8 v[130:131], v159 offset:33792")
        AL_("global_load_dwordx4 v[92:95], v160, %[abase]")
        "s_waitcnt lgkmcnt(8) vmcnt(7)\n\t"
        MF_("v_mfma_i32_32x32x32_i8 %[c0], v[96:99], v[132:135], %[c0]")
        "ds_read2_b32 v[148:149], v151 offset0:3 offset1:12\n\t"
        "v_add_u32 v150, -1, %[thr]\n\t"
        "ds_write_b32 %[cons], v150\n\t"
        RD_("ds_read_b64_tr_b8 v[132:133], v156 offset:36864")
        RD_("ds_read_b64_tr_b8 v[134:135], v156 offset:37888")
        MF_("v_mfma_i32_32x32x32_i8 %[c1], v[96:99], v[136:139], %[c1]")
        RD_("ds_read_b64_tr_b8 v[136:137], v157 offset:36864")
        RD_("ds_read_b64_tr_b8 v[138:139], v157 offset:37888")
        MF_("v_mfma_i32_32x32x32_i8 %[c2], v[96:99], v[140:143], %[c2]")
        RD_("ds_read_b64_tr_b8 v[140:141], v158 offset:36864")
        RD_("ds_read_b64_tr_b8 v[142:143], v158 offset:37888")
        MF_("v_mfma_i32_32x32x32_i8 %[c3], v[96:99], v[144:147], %[c3]")
        RD_("ds_read_b64_tr_b8 v[144:145], v159 offset:36864")
        RD_("ds_read_b64_tr_b8 v[146:147], v159 offset:37888")
        AL_("global_load_dwordx4 v[96:99], v160, %[abase] offset:1024")
        "s_add_u32 %[thr], %[thr], 1\n\t"
        "s_waitcnt lgkmcnt(8) vmcnt(7)\n\t"
        MF_("v_mfma_i32_32x32x32_i8 %[c0], v[100:103], v[116:119], %[c0]")
        "v_min_i32 v150, v148, v149\n\t"
        "v_readlane_b32 %[t], v161, 10\n\t"
        "v_cmp_gt_i32 vcc, %[thr], v150\n\t"
        MF_("v_mfma_i32_32x32x32_i8 %[c1], v[100:103], v[120:123], %[c1]")
        "v_add_u32 v160, %[t], %[voff]\n\t"
        "s_cbranch_vccnz 70f\n\t"
        "71:\n\t"
        RD_("ds_read_b64_tr_b8 v[116:117], v156 offset:49152")
        RD_("ds_read_b64_tr_b8 v[118:119], v156 offset:50176")
        RD_("ds_read_b64_tr_b8 v[120:121], v157 offset:49152")
        RD_("ds_read_b64_tr_b8 v[122:123], v157 offset:50176")
        MF_("v_mfma_i32_32x32x32_i8 %[c2], v[100:103], v[124:127], %[c2]")
        RD_("ds_read_b64_tr_b8 v[124:125], v158 offset:49152")
        RD_("ds_read_b64_tr_b8 v[126:127], v158 offset:50176")
        MF_("v_mfma_i32_32x32x32_i8 %[c3], v[100:103], v[128:131], %[c3]")
        RD_("ds_read_b64_tr_b8 v[128:129], v159 offset:49152")
        RD_("ds_read_b64_tr_b8 v[130:131], v159 offset:50176")
        AL_("global_load_dwordx4 v[100:103], v160, %[abase]")
        "s_waitcnt lgkmcnt(8) vmcnt(7)\n\t"
        MF_("v_mfma_i32_32x32x32_i8 %[c0], v[104:107], v[132:135], %[c0]")
        "ds_read2_b32 v[148:149], v151 offset0:0 offset1:12\n\t"
        "v_add_u32 v150, -1, %[thr]\n\t"
        "ds_write_b32 %[cons], v150\n\t"
        RD_("ds_read_b64_tr_b8 v[132:133], v156 offset:53248")
        RD_("ds_read_b64_tr_b8 v[134:135], v156 offset:54272")
        MF_("v_mfma_i32_32x32x32_i8 %[c1], v[104:107], v[136:139], %[c1]")
        RD_("ds_read_b64_tr_b8 v[136:137], v157 offset:53248")
        RD_("ds_read_b64_tr_b8 v[138:139], v157 offset:54272")
        MF_("v_mfma_i32_32x32x32_i8 %[c2], v[104:107], v[140:143], %[c2]")
        RD_("ds_read_b64_tr_b8 v[140:141], v158 offset:53248")
        RD_("ds_read_b64_tr_b8 v[142:143], v158 offset:54272")
        MF_("v_mfma_i32_32x32x32_i8 %[c3], v[104:107], v[144:147], %[c3]")
        RD_("ds_read_b64_tr_b8 v[144:145], v159 offset:53248")
        RD_("ds_read_b64_tr_b8 v[146:147], v159 offset:54272")
        AL_("global_load_dwordx4 v[104:107], v160, %[abase] offset:1024")
        "s_add_u32 %[thr], %[thr], 1\n\t"
        "s_waitcnt lgkmcnt(8) vmcnt(7)\n\t"
        MF_("v_mfma_i32_32x32x32_i8 %[c0], v[108:111], v[116:119], %[c0]")
        "v_min_i32 v150, v148, v149\n\t"
        "v_readlane_b32 %[t], v161, 11\n\t"
        "v_cmp_gt_i32 vcc, %[thr], v150\n\t"
        MF_("v_mfma_i32_32x32x32_i8 %[c1], v[108:111], v[120:123], %[c1]")
        "v_add_u32 v160, %[t], %[voff]\n\t"
        "s_cbranch_vccnz 80f\n\t"
        "81:\n\t"
        RD_("ds_read_b64_tr_b8 v[116:117], v152 offset:0")
        RD_("ds_read_b64_tr_b8 v[118:119], v152 offset:1024")
        RD_("ds_read_b64_tr_b8 v[120:121], v153 offset:0")
        RD_("ds_read_b64_tr_b8 v[122:123], v153 offset:1024")
        MF_("v_mfma_i32_32x32x32_i8 %[c2], v[108:111], v[124:127], %[c2]")
        RD_("ds_read_b64_tr_b8 v[124:125], v154 offset:0")
        RD_("ds_read_b64_tr_b8 v[126:127], v154 offset:1024")
        MF_("v_mfma_i32_32x32x32_i8 %[c3], v[108:111], v[128:131], %[c3]")
        RD_("ds_read_b64_tr_b8 v[128:129], v155 offset:0")
        RD_("ds_read_b64_tr_b8 v[130:131], v155 offset:1024")
        AL_("global_load_dwordx4 v[108:111], v160, %[abase]")
        "s_waitcnt lgkmcnt(8) vmcnt(7)\n\t"
        MF_("v_mfma_i32_32x32x32_i8 %[c0], v[112:115], v[132:135], %[c0]")
        "ds_read2_b32 v[148:149], v151 offset0:1 offset1:12\n\t"
        "v_add_u32 v150, -1, %[thr]\n\t"
        "ds_write_b32 %[cons], v150\n\t"
        RD_("ds_read_b64_tr_b8 v[132:133], v152 offset:4096")
        RD_("ds_read_b64_tr_b8 v[134:135], v152 offset:5120")
        MF_("v_mfma_i32_32x32x32_i8 %[c1], v[112:115], v[136:139], %[c1]")
        RD_("ds_read_b64_tr_b8 v[136:137], v153 offset:4096")
        RD_("ds_read_b64_tr_b8 v[138:139], v153 offset:5120")
        MF_("v_mfma_i32_32x32x32_i8 %[c2], v[112:115], v[140:143], %[c2]")
        RD_("ds_read_b64_tr_b8 v[140:141], v154 offset:4096")
        RD_("ds_read_b64_tr_b8 v[142:143], v154 offset:5120")
        MF_("v_mfma_i32_32x32x32_i8 %[c3], v[112:115], v[144:147], %[c3]")
        RD_("ds_read_b64_tr_b8 v[144:145], v155 offset:4096")
        RD_("ds_read_b64_tr_b8 v[146:147], v155 offset:5120")
        AL_("global_load_dwordx4 v[112:115], v160, %[abase] offset:1024")
        CNT_("s_memrealtime s[100:101]")
        CNT_("s_waitcnt lgkmcnt(0)")
        CNT_("s_lshr_b32 %[t], %[thr], 3")
        CNT_("s_mov_b32 m0, %[t]")
        CNT_("s_nop 1")
        CNT_("v_writelane_b32 %[tl], s100, m0")
        CNT_("s_memtime s[100:101]")
        CNT_("s_add_u32 m0, m0, 8")
        CNT_("s_waitcnt lgkmcnt(0)")
        CNT_("v_writelane_b32 %[tl], s100, m0")
        "ds_bpermute_b32 v161, %[rot], v161\n\t"
        "s_cmp_lt_u32 %[thr], 33\n\t"
        "s_waitcnt lgkmcnt(0)\n\t"
        "s_cbranch_scc1 1b\n\t"
        "s_branch 9f\n\t"
        "10:\n\t"
        "s_sleep 1\n\t"
        CNT_("s_add_u32 %[n1], %[n1], 1")
        "ds_read2_b32 v[148:149], v151 offset0:1 offset1:12\n\t"
        "s_waitcnt lgkmcnt(0)\n\t"
        "v_min_i32 v150, v148, v149\n\t"
        "v_cmp_gt_i32 vcc, %[thr], v150\n\t"
        "s_nop 4\n\t"
        "s_cbranch_vccnz 10b\n\t"
        "s_branch 11b\n\t"
        "20:\n\t"
        "s_sleep 1\n\t"
        CNT_("s_add_u32 %[n1], %[n1], 1")
        "ds_read2_b32 v[148:149], v151 offset0:2 offset1:12\n\t"
        "s_waitcnt lgkmcnt(0)\n\t"
        "v_min_i32 v150, v148, v149\n\t"
        "v_cmp_gt_i32 vcc, %[thr], v150\n\t"
        "s_nop 4\n\t"
        "s_cbranch_vccnz 20b\n\t"
        "s_branch 21b\n\t"
        "30:\n\t"
        "s_sleep 1\n\t"
        CNT_("s_add_u32 %[n1], %[n1], 1")
        "ds_read2_b32 v[148:149], v151 offset0:3 offset1:12\n\t"
        "s_waitcnt lgkmcnt(0)\n\t"
        "v_min_i32 v150, v148, v149\n\t"
        "v_cmp_gt_i32 vcc, %[thr], v150\n\t"
        "s_nop 4\n\t"
        "s_cbranch_vccnz 30b\n\t"
        "s_branch 31b\n\t"
        "40:\n\t"
        "s_sleep 1\n\t"
        CNT_("s_add_u32 %[n1], %[n1], 1")
        "ds_read2_b32 v[148:149], v151 offset0:0 offset1:12\n\t"
        "s_waitcnt lgkmcnt(0)\n\t"
        "v_min_i32 v150, v148, v149\n\t"
        "v_cmp_gt_i32 vcc, %[thr], v150\n\t"
        "s_nop 4\n\t"
        "s_cbranch_vccnz 40b\n\t"
        "s_branch 41b\n\t"
        "50:\n\t"
        "s_sleep 1\n\t"
        CNT_("s_add_u32 %[n1], %[n1], 1")
        "ds_read2_b32 v[148:149], v151 offset0:1 offset1:12\n\t"
        "s_waitcnt lgkmcnt(0)\n\t"
        "v_min_i32 v150, v148, v149\n\t"
        "v_cmp_gt_i32 vcc, %[thr], v150\n\t"
        "s_nop 4\n\t"
        "s_cbranch_vccnz 50b\n\t"
        "s_branch 51b\n\t"
        "60:\n\t"
        "s_sleep 1\n\t"
        CNT_("s_add_u32 %[n1], %[n1], 1")
        "ds_read2_b32 v[148:149], v151 offset0:2 offset1:12\n\t"
        "s_waitcnt lgkmcnt(0)\n\t"
        "v_min_i32 v150, v148, v149\n\t"
        "v_cmp_gt_i32 vcc, %[thr], v150\n\t"
        "s_nop 4\n\t"
        "s_cbranch_vccnz 60b\n\t"
        "s_branch 61b\n\t"
        "70:\n\t"
        "s_sleep 1\n\t"
        CNT_("s_add_u32 %[n1], %[n1], 1")
        "ds_read2_b32 v[148:149], v151 offset0:3 offset1:12\n\t"
        "s_waitcnt lgkmcnt(0)\n\t"
        "v_min_i32 v150, v148, v149\n\t"
        "v_cmp_gt_i32 vcc, %[thr], v150\n\t"
        "s_nop 4\n\t"
        "s_cbranch_vccnz 70b\n\t"
        "s_branch 71b\n\t"
        "80:\n\t"
        "s_sleep 1\n\t"
        CNT_("s_add_u32 %[n1], %[n1], 1")
        "ds_read2_b32 v[148:149], v151 offset0:0 offset1:12\n\t"
        "s_waitcnt lgkmcnt(0)\n\t"
        "v_min_i32 v150, v148, v149\n\t"
        "v_cmp_gt_i32 vcc, %[thr], v150\n\t"
        "s_nop 4\n\t"
        "s_cbranch_vccnz 80b\n\t"
        "s_branch 81b\n\t"
        "9:\n\t"
        "s_waitcnt lgkmcnt(0) vmcnt(0)\n\t"
        "s_nop 15\n\t"
        "s_nop 15\n\t"
        : [c0] "+v"(c0), [c1] "+v"(c1), [c2] "+v"(c2), [c3] "+v"(c3), [thr] "=&s"(sthr), [t] "=&s"(st)
        : [tb0] "v"(tb0), [tb1] "v"(tb1), [tb2] "v"(tb2), [tb3] "v"(tb3), [tab] "v"(tab), [voff] "v"(voff), [abase] "s"(abase),
          [landb] "s"(L_LANDB), [cons] "v"(L_CONS + cw * 4), [rot] "v"((unsigned)(((lane + 8) & 63) * 4))
        : "memory", "scc", "vcc", "s100", "s101", "v84", "v85", "v86", "v87", "v88", "v89", "v90", "v91", "v92", "v93", "v94", "v95", "v96", "v97", "v98", "v99", "v100", "v101", "v102", "v103", "v104", "v105", "v106", "v107", "v108", "v109", "v110", "v111", "v112", "v113", "v114", "v115", "v116", "v117", "v118", "v119", "v120", "v121", "v122", "v123", "v124", "v125", "v126", "v127", "v128", "v129", "v130", "v131", "v132", "v133", "v134", "v135", "v136", "v137", "v138", "v139", "v140", "v141", "v142", "v143", "v144", "v145", "v146", "v147", "v148", "v149", "v150", "v151", "v152", "v153", "v154", "v155", "v156", "v157", "v158", "v159", "v160", "v161");
#undef MF_
#undef CNT_
#undef AL_
#undef RD_
    }
    while (lds_min4(L_CONS) < 28) __builtin_amdgcn_s_sleep(1);
    {
        const unsigned xs = lds0 + LDS_B + (unsigned)((26 % NB) * 16384), xs2 = lds0 + LDS_B + (unsigned)((27 % NB) * 16384);
        const unsigned mine = (kh ? xs2 : xs) + (unsigned)(mh * 8192 + lane * 16);
        const unsigned theirs = (kh ? xs : xs2) + (unsigned)(mh * 8192 + lane * 16);
        const v16i s0 = c2, s1 = c3;
#pragma unroll
        for (int i = 0; i < 4; ++i) {
            *(LAS v4i*)(uintptr_t)(mine + i * 1024) = (v4i){s0[4 * i], s0[4 * i + 1], s0[4 * i + 2], s0[4 * i + 3]};
            *(LAS v4i*)(uintptr_t)(mine + 4096 + i * 1024) = (v4i){s1[4 * i], s1[4 * i + 1], s1[4 * i + 2], s1[4 * i + 3]};
        }
        asm volatile("s_waitcnt lgkmcnt(0)" ::: "memory");
        if (lane == 0) lds_put(L_CONS + cw * 4, 64);
        while (lds_get(L_CONS + (cw ^ 2) * 4) < 64) __builtin_amdgcn_s_sleep(1);
        asm volatile("" ::: "memory");
#pragma unroll
        for (int i = 0; i < 4; ++i) {
            v4i u = *(LAS v4i*)(uintptr_t)(theirs + i * 1024);
            v4i v = *(LAS v4i*)(uintptr_t)(theirs + 4096 + i * 1024);
            c0[4 * i] += u.x; c0[4 * i + 1] += u.y; c0[4 * i + 2] += u.z; c0[4 * i + 3] += u.w;
            c1[4 * i] += v.x; c1[4 * i + 1] += v.y; c1[4 * i + 2] += v.z; c1[4 * i + 3] += v.w;
        }
    }
    const bool bad = *(volatile LAS int*)(uintptr_t)L_BAD != 0;

    const float scale = bad ? __builtin_nanf("") : scale0;
    float* optr = P.out + (size_t)(mb * 64 + mh * 32 + 4 * (lane >> 5)) * NN + nb * 128 + kh * 64 + (lane & 31);
    typedef float v2f __attribute__((ext_vector_type(2)));
#pragma unroll
    for (int r = 0; r < 16; ++r) {
        const int row = (r & 3) + 8 * (r >> 2);
        const v2f y = (v2f){(float)c0[r], (float)c1[r]} * scale;
        v2f e = (v2f){__builtin_amdgcn_fmed3f(y.x, -60.f, 60.f), __builtin_amdgcn_fmed3f(y.y, -60.f, 60.f)} * -1.44269504088896340736f;
        e = (v2f){__builtin_amdgcn_exp2f(e.x), __builtin_amdgcn_exp2f(e.y)} + 1.0f;
        const v2f o = y * (v2f){__builtin_amdgcn_rcpf(e.x), __builtin_amdgcn_rcpf(e.y)};
        const float o0 = o.x, o1 = o.y;
        __hip_atomic_store(optr + (size_t)row * NN, o0, __ATOMIC_RELAXED, __HIP_MEMORY_SCOPE_AGENT);
        __hip_atomic_store(optr + (size_t)row * NN + 32, o1, __ATOMIC_RELAXED, __HIP_MEMORY_SCOPE_AGENT);
    }
}

}

extern "C" void kernel_launch(int rank, int nranks, ncclComm_t comm,
                              hipStream_t stream,
                              void* const* d_in, const int* in_sizes, int n_in,
                              void* d_out, int out_size,
                              const int* in_dtypes, int out_dtype,
                              void* const* d_out_all,
                              void* const* d_ws_all, size_t ws_bytes) {
    if (nranks != WORLD || ws_bytes < WS_NEED || (unsigned)rank >= (unsigned)WORLD) {
        fprintf(stderr, "unsupported configuration\n");
        return;
    }
    static bool attr_set[WORLD] = {};
    if (!attr_set[rank]) {
        HIP_CHECK(hipFuncSetAttribute((const void*)fused_kernel, hipFuncAttributeMaxDynamicSharedMemorySize, LDS_BYTES));
        attr_set[rank] = true;
    }
    Params P;
    P.x = (const signed char*)d_in[0];
    P.w = (const signed char*)d_in[1];
    P.sx = (const float*)d_in[2];
    P.sw = (const float*)d_in[3];
    P.out = (float*)d_out;
    for (int p = 0; p < WORLD; ++p) P.ws[p] = (char*)d_ws_all[p];
    P.rank = rank;
    hipLaunchKernelGGL(fused_kernel, dim3(NWG), dim3(NTHR), LDS_BYTES, stream, P);
}
```
